```python
import numpy as np

kernel_name = "gpu_gemm_rs_m4096_k4096_n4096_w8a8_dqrelu"


ABSMAX_RTOL = 0.001

COMPUTE_DTYPE = "int8"
EPILOGUE = "dequant_relu"
IN_QUANT = "int8"
OUT_QUANT = None
IN_DTYPES = {"x": "int8", "w_mat": "int8",
             "scale_x": "float32", "scale_w": "float32"}
WORLD_SIZE = 8

def setup_inputs():

    x = np.random.randint(-127, 128, size=(4096, 4096)).astype(np.int8)
    w_mat = np.random.randint(-127, 128, size=(4096, 4096)).astype(np.int8)
    scale_x = (0.02 * (1.0 + np.random.rand(1))).astype(np.float32)
    scale_w = (0.02 * (1.0 + np.random.rand(1))).astype(np.float32)
    return {"x": x, "w_mat": w_mat, "scale_x": scale_x, "scale_w": scale_w}

def reference(x, w_mat, scale_x, scale_w):
    acc = x.astype(np.float64) @ w_mat.astype(np.float64)
    y = (acc.astype(np.float32) * (scale_x * scale_w)).astype(np.float32)
    return np.maximum(y, 0.0).astype(np.float32)
```

```cpp
#include <hip/hip_runtime.h>
#include <rccl/rccl.h>
#include <cstdio>
#include <cstring>
#include <cstdlib>

typedef unsigned u32;
typedef unsigned long long u64;
typedef __attribute__((ext_vector_type(4))) unsigned u32x4;
typedef __attribute__((ext_vector_type(2))) unsigned u32x2;
typedef __attribute__((ext_vector_type(4))) int i32x4;
typedef __attribute__((ext_vector_type(16))) int i32x16;
typedef __attribute__((ext_vector_type(4))) float f32x4;
typedef __attribute__((address_space(3))) u32* lds_u32p;
typedef __attribute__((address_space(3))) u32x4* lds_u32x4p;
typedef __attribute__((address_space(1))) u32* glb_u32p;

constexpr u32 KiB = 1024, MiB = 1024 * 1024;
constexpr int NRANK = 8;
constexpr u32 XB = 256 * KiB;
constexpr u32 WB = 2 * MiB;
constexpr u32 BS = 16384 + 1024;
constexpr u32 RS = 16 * BS;
constexpr u32 XREG = 0;
constexpr u32 WREG = 8 * RS;
constexpr u32 SLOT = 16 * RS;
constexpr u32 CTRL = 64 * MiB;
constexpr u32 XFLAG = CTRL;
constexpr u32 WFLAG = CTRL + 64 * KiB;
constexpr u32 WCNT = CTRL + 96 * KiB;
constexpr u32 ORG = CTRL + 128 * KiB;
constexpr u32 INVW = CTRL + 192 * KiB;
constexpr u32 CTRS = CTRL + 256 * KiB;
constexpr u32 TMO = CTRL + 384 * KiB;
constexpr u32 DBG = CTRL + 512 * KiB;
constexpr u32 WS_USED = 65 * MiB;
constexpr u32 OUT_BYTES = 8 * MiB;
constexpr u32 DEADLINE = 50000000u;

struct Params {
  char* pws[NRANK];
  const char* x; const char* w; const float* sx; const float* sw; float* out;
  int rank;
  int taux, tauw;
  int burst;
  int first;
  int cfirst;
  int minorg;
  int godelay;
  int bdelay, cdelay, cstag;
  int tauself;
  int lag;
  int lagend;
};

__device__ __forceinline__ __amdgpu_buffer_rsrc_t mkrsrc(const void* p, u32 bytes) {
  return __builtin_amdgcn_make_buffer_rsrc((void*)p, 0, bytes, 0x00020000);
}
template <int N> __device__ __forceinline__ void waitv() { asm volatile("s_waitcnt vmcnt(%0)" ::"n"(N) : "memory"); }
__device__ __forceinline__ void waitl0() { asm volatile("s_waitcnt lgkmcnt(0)" ::: "memory"); }
__device__ __forceinline__ u32 lds_ld(u32 a) { return __hip_atomic_load((lds_u32p)(size_t)a, __ATOMIC_RELAXED, __HIP_MEMORY_SCOPE_WORKGROUP); }
__device__ __forceinline__ void lds_st(u32 a, u32 v) { __hip_atomic_store((lds_u32p)(size_t)a, v, __ATOMIC_RELAXED, __HIP_MEMORY_SCOPE_WORKGROUP); }
__device__ __forceinline__ u32x4 rawrsrc(const void* p, u32 bytes) {
  const u64 a = (u64)p;
  u32x4 r = {(u32)a, (u32)(a >> 32) & 0xffffu, bytes, 0x00020000u};
  return r;
}
#define ALOAD(dst, r, voff, imm) asm volatile("buffer_load_dwordx4 %0, %1, %2, 0 offen offset:%3" : "+v"(dst) : "v"(voff), "s"(r), "n"(imm) : "memory")
#define ALOADW(dst, r, voff) asm volatile("buffer_load_dword %0, %1, %2, 0 offen sc0 sc1" : "+v"(dst) : "v"(voff), "s"(r) : "memory")
#define AWAIT1(n, x) asm volatile("s_waitcnt vmcnt(%1)" : "+v"(x) : "n"(n) : "memory")
__device__ __forceinline__ u32 now32() { return (u32)__builtin_amdgcn_s_memrealtime(); }

#define GOIDX 8
#define DIST 4
#define PWIN 4
#define OUTAUX 17
#define INV 1

__device__ __forceinline__ bool wait_words(__amdgpu_buffer_rsrc_t rme, u32 off, u32 want, u32 call, u32 t0, int lane) {
  bool ok = true;
  while (true) {
    const u32 tp = now32();
    const u32 f = __builtin_amdgcn_raw_buffer_load_b32(rme, off, 0, 17);
    if (__all((int)(f - want) >= 0)) break;
    if (__builtin_expect(tp - t0 > 100000u, 0)) {
      const u32 tm = __builtin_amdgcn_raw_buffer_load_b32(rme, TMO, 0, 16);
      if (__builtin_amdgcn_readfirstlane(tm) == call || tp - t0 > DEADLINE) {
        if (lane == 0) __builtin_amdgcn_raw_buffer_store_b32(call, rme, TMO, 0, 16);
        ok = false;
        break;
      }
      __builtin_amdgcn_s_sleep(32);
    }
    __builtin_amdgcn_s_sleep(1);
  }
  asm volatile("" ::: "memory");
  return ok;
}

__device__ __forceinline__ u32 elect_invalidate(char* me, __amdgpu_buffer_rsrc_t rme, u32 call, int lane, bool vote) {
  u32 xcc; asm volatile("s_getreg_b32 %0, hwreg(HW_REG_XCC_ID)" : "=s"(xcc));
  const u32 invw = INVW + (xcc & 15) * 512;
  if (INV && vote) {
    u32 old = call;
    if (lane == 0) old = __hip_atomic_exchange((glb_u32p)(me + invw), call, __ATOMIC_RELAXED, __HIP_MEMORY_SCOPE_AGENT);
    if (__builtin_amdgcn_readfirstlane(old) != call) {
      __builtin_amdgcn_fence(__ATOMIC_ACQUIRE, "");
      waitv<0>();
      if (lane == 0) __builtin_amdgcn_raw_buffer_store_b32(call, rme, invw + 256, 0, 16);
    }
  }
  return invw + 256;
}

__global__ void __launch_bounds__(512) fused(Params P) {
  __shared__ __attribute__((aligned(1024))) char smem[96 * KiB + 128];
  const int b = blockIdx.x, t = threadIdx.x;
  const int lane = t & 63;
  const int wv = __builtin_amdgcn_readfirstlane(t >> 6);
  const int g = b & 7, cu = b >> 3;
  const int rank = P.rank;
  auto rme = mkrsrc(P.pws[0], WS_USED);
  const u32 lds0 = (u32)(size_t)(__attribute__((address_space(3))) char*)smem;
  const u32 ldc = lds0 + 96 * KiB;
  const u32 ctr = CTRS + (b * 8 + wv) * 4;
  const u32 callv = __builtin_amdgcn_raw_buffer_load_b32(rme, ctr, 0, 16);

  if (__builtin_expect(g != 0 && wv == 0 && cu < 16, 1)) {
      const int s = cu;
      const int peer = (rank + g) & 7;
      auto rp = mkrsrc(P.pws[g], WS_USED);
      auto rx = mkrsrc(P.x, WB);
      u32x4 xv[16];
#pragma unroll
      for (int i = 0; i < 16; ++i) xv[i] = __builtin_amdgcn_raw_buffer_load_b128(rx, peer * XB + s * 16384 + (lane & 31) * 512 + (lane >> 5) * 16 + i * 32, 0, 0);
      const u32 call = __builtin_amdgcn_readfirstlane(callv) + 1;
      const u32 dst = (call & 1) * SLOT + XREG + rank * RS + s * BS + lane * 16;
      const u32 fl = XFLAG + (rank * 16 + s) * 512;
      const int tau = P.taux;
      asm volatile("" :: "v"(xv[0]));
      const u32 org = now32();
      int due = -(P.burst * tau >> 2) + s * tau / 16;
#pragma unroll
      for (int i = 0; i < 16; ++i) {
        while ((int)((now32() - org) << 8) < due) __builtin_amdgcn_s_sleep(1);
        due += tau;
        if (i == GOIDX) { if (lane == 0) __builtin_amdgcn_raw_buffer_store_b32(call, rp, fl + 256, 0, 17); }
        __builtin_amdgcn_raw_buffer_store_b128(xv[i], rp, dst + i * 1024, 0, 17);
      }
      waitv<0>();
      if (lane == 0) __builtin_amdgcn_raw_buffer_store_b32(call, rp, fl, 0, 17);
    }
  if (cu >= 16) {
    u32 call;
    const int p = cu - 16;
    const u32 t0 = now32();
    int task = -1;
    if (g == 0 && (wv == 2 || wv == 3)) task = p * 2 + (wv & 1);
    else if (p >= 8) {
      if (wv >= 5) task = 64 + (((p - 8) * 8 + g) * 3 + wv - 5);
      else if (g == 0) task = 32 + (p - 8) * 4 + (wv == 4 ? 2 : wv) + 0;
    }
    if (g == 0 && p == 7 && task < 0) task = 32 + (wv < 2 ? wv : wv - 2) * 4 + 3;
    if (g == 0 && p == 6 && wv >= 6) task = 32 + wv * 4 + 3;
    if (task >= 0) {
      const int ph = task >> 5, blk = (task >> 2) & 7, cq = task & 3;
      const int stage = ph < 2 ? ph : 2;
      char* pw = P.pws[lane & 7];
      auto rw = mkrsrc(P.w, WB);
      const int kq = lane >> 4, q = lane & 15;
      if (stage) { const int at = stage == 1 ? P.bdelay : P.cdelay + (p - 8) * P.cstag; while ((int)(now32() - t0) < at) __builtin_amdgcn_s_sleep(4); }
      const u32 src = (u32)(64 * blk + 16 * kq) * 4096u + 512 * ph + 128 * cq + 8 * q;
      u32x2 in[16];
#pragma unroll
      for (int r = 0; r < 16; ++r) in[r] = __builtin_amdgcn_raw_buffer_load_b64(rw, src, r * 4096, 0);
      asm volatile("" : "+v"(pw));
      call = __builtin_amdgcn_readfirstlane(callv) + 1;
      u32x4 o[8];
#pragma unroll
      for (int d = 0; d < 2; ++d)
#pragma unroll
        for (int m = 0; m < 4; ++m) {
          u32 a0 = in[4 * m][d], a1 = in[4 * m + 1][d], a2 = in[4 * m + 2][d], a3 = in[4 * m + 3][d];
          u32 t0_ = __builtin_amdgcn_perm(a1, a0, 0x05010400), t1_ = __builtin_amdgcn_perm(a1, a0, 0x07030602);
          u32 t2_ = __builtin_amdgcn_perm(a3, a2, 0x05010400), t3_ = __builtin_amdgcn_perm(a3, a2, 0x07030602);
          o[4 * d + 0][m] = __builtin_amdgcn_perm(t2_, t0_, 0x05040100);
          o[4 * d + 1][m] = __builtin_amdgcn_perm(t2_, t0_, 0x07060302);
          o[4 * d + 2][m] = __builtin_amdgcn_perm(t3_, t1_, 0x05040100);
          o[4 * d + 3][m] = __builtin_amdgcn_perm(t3_, t1_, 0x07060302);
        }
      const u32 lw = lds0 + wv * 8192;
      const u32 wbase = lw + (q >> 2) * 2048 + (kq >> 1) * 1024 + (8 * (q & 3) + 32 * (kq & 1)) * 16;
#pragma unroll
      for (int e = 0; e < 8; ++e) *(lds_u32x4p)(size_t)(wbase + ((e ^ (q & 7)) << 4)) = o[e];
      waitl0();
      const u32 ra = lw + (((lane & ~7) | ((lane & 7) ^ ((lane >> 3) & 3))) << 4);
      const u32 dst = (call & 1) * SLOT + WREG + ph * RS + cq * 4 * BS + blk * 2048 + lane * 16;
#pragma unroll
      for (int i = 0; i < 8; ++i) o[i] = *(lds_u32x4p)(size_t)((ra ^ (((i >> 1) & 1) * 64)) + i * 1024);
#pragma unroll
      for (int i = 0; i < 8; ++i) __builtin_amdgcn_raw_buffer_store_b128(o[i], rme, dst + (i >> 1) * BS + (i & 1) * 1024, 0, 17);
      waitv<0>();
      u32 old = 0;
      if (lane == 0) old = __hip_atomic_fetch_add((glb_u32p)(P.pws[0] + WCNT + stage * 256), 1u, __ATOMIC_RELAXED, __HIP_MEMORY_SCOPE_AGENT);
      old = __builtin_amdgcn_readfirstlane(old);
      if (old + 1 == call * (stage < 2 ? 32u : 192u) && lane < NRANK)
        __hip_atomic_store((glb_u32p)(pw + WFLAG + (rank * 4 + stage) * 256), call, __ATOMIC_RELAXED, __HIP_MEMORY_SCOPE_SYSTEM);
    }
    call = __builtin_amdgcn_readfirstlane(callv) + 1;
    if (lane == 0) __builtin_amdgcn_raw_buffer_store_b32(call, rme, ctr, 0, 16);
    if (p < 7 && wv < 2) {
      const int gs = p + 1, src = (rank + gs) & 7, c = 2 * g + wv;
      const u32x4 rs = rawrsrc(P.pws[gs], WS_USED);
      const u32x4 rm = rawrsrc(P.pws[0], WS_USED);
      const u32 invd = elect_invalidate(P.pws[0], rme, call, lane, true);
      const u32 wf = WFLAG + src * 1024;
      while ((int)(now32() - t0) < P.first) __builtin_amdgcn_s_sleep(4);
      bool ok = true;
      u32 org = t0 + P.minorg;
#pragma unroll 1
      for (int k = 0; k < 2 && ok; ++k) {
        ok = wait_words(rme, k == 0 ? XFLAG + (src * 16 + c) * 512 + 256 : (INV && lane == 1) ? invd : wf, call, call, t0, lane);
        const u32 cand = now32() + (k == 0 ? P.godelay : 0);
        if ((int)(cand - org) > 0) org = cand;
      }
      if (lane == 0) { u32x2 o = {call, org}; __builtin_amdgcn_raw_buffer_store_b64(o, rme, ORG + (gs * 16 + c) * 256, 0, 16); }
      int due = 0;
      const int tau = P.tauw;
      u32 wj = (call & 1) * SLOT + WREG + c * BS + lane * 16;
      u32x4 v[4];
      u32 pk = 0;
#pragma unroll
      for (int i = 0; i < 4; ++i) v[i] = u32x4{0, 0, 0, 0};
#pragma unroll 1
      for (int q = 0; q < 32 && ok; ++q) {
        if (q == 2 || q == 6) { const u32 vo = wf + (q == 2 ? 256 : 512); ALOADW(pk, rm, vo); }
        if (q == 4 || q == 8) {
          asm volatile("" : "+v"(pk));
          if (!__all((int)(pk - call) >= 0)) ok = wait_words(rme, wf + (q == 4 ? 256 : 512), call, call, t0, lane);
          if (!ok) break;
        }
#pragma unroll
        for (int i = 0; i < 4; ++i) {
          while ((int)((now32() - org) << 8) < due) __builtin_amdgcn_s_sleep(1);
          due += tau;
          waitv<PWIN - 1>();
          ALOAD(v[i], rs, wj, i * 1024);
        }
        wj += (q & 3) == 3 ? RS - 3 * 4096 : 4096;
      }
      waitv<0>();
    }
  } else {
    const u32 t0 = now32();
    {
      u32x4 z = {0, 0, 0, 0};
#pragma unroll
      for (int i = 0; i < 8; ++i) *(lds_u32x4p)(size_t)(lds0 + wv * 8192 + i * 1024 + lane * 16) = z;
      if (wv == 0 && lane < 16) *(lds_u32p)(size_t)(ldc + lane * 4) = 0;
    }
    const u32 call = __builtin_amdgcn_readfirstlane(callv) + 1;
    if (lane == 0) __builtin_amdgcn_raw_buffer_store_b32(call, rme, ctr, 0, 16);
    const float scale = P.sx[0] * P.sw[0];
    const u32 invd = elect_invalidate(P.pws[0], rme, call, lane, wv == 1);
    waitl0();
    __builtin_amdgcn_s_barrier();
    asm volatile("" ::: "memory");
    const int gs = wv;
    const int src = (rank + gs) & 7;
    const int c = 2 * g + (cu & 1), rg = cu >> 1;
    const u32x4 rs = rawrsrc(P.pws[gs], WS_USED), rm = rawrsrc(P.pws[0], WS_USED);
    const u32 slot = (call & 1) * SLOT;
    const u32 wf = WFLAG + src * 1024;
    const u32 og = ORG + (gs * 16 + c) * 256;
    bool ok;
    i32x4 a0[16], a1[16], bf[8];
    u32 org;
    {
      u32 off = wf;
      if (gs) {
        while ((int)(now32() - t0) < P.cfirst) __builtin_amdgcn_s_sleep(4);
        off = lane == 0 ? wf : lane == 1 ? og : (INV && lane == 2) ? invd : XFLAG + (src * 16 + 2 * rg + (lane & 1)) * 512;
      }
      ok = wait_words(rme, off, call, call, t0, lane);
      const u32 ov = __builtin_amdgcn_raw_buffer_load_b32(rme, og + 4, 0, 17);
      u32 xo = gs ? slot + XREG + src * RS + 2 * rg * BS + lane * 16 : rank * XB + 2 * rg * 16384 + (lane & 31) * 512 + (lane >> 5) * 16;
      const u32 xs1 = gs ? BS : 16384, xsf = gs ? 1024 : 32;
      auto ra = gs ? rme : mkrsrc(P.x, WB);
#pragma unroll
      for (int f = 0; f < 16; ++f) {
        a0[f] = (i32x4)__builtin_amdgcn_raw_buffer_load_b128(ra, xo, 0, 0);
        a1[f] = (i32x4)__builtin_amdgcn_raw_buffer_load_b128(ra, xo + xs1, 0, 0);
        xo += xsf;
      }
      org = gs && ok ? __builtin_amdgcn_readfirstlane(ov) : now32() - P.lag;
      waitv<0>();
#pragma unroll
      for (int f = 0; f < 16; ++f) asm volatile("" : "+v"(a0[f]), "+v"(a1[f]));
    }
    if (!ok && lane == 0) lds_st(ldc + 32, 1);
    int due = P.lag * 256;
    const int tau = gs ? P.tauw - (P.lag - P.lagend) * 2 : P.tauself;
#define PACE() { while ((int)((now32() - org) << 8) < due) __builtin_amdgcn_s_sleep(1); due += tau; }
    i32x16 acc0 = {0, 0, 0, 0, 0, 0, 0, 0, 0, 0, 0, 0, 0, 0, 0, 0}, acc1 = acc0;
    u32 wj = slot + WREG + c * BS + lane * 16;
    auto rout = mkrsrc(P.out, OUT_BYTES);
#define MMA(f) { acc0 = __builtin_amdgcn_mfma_i32_32x32x32_i8(a0[f], bf[(f) & 7], acc0, 0, 0, 0); \
                 acc1 = __builtin_amdgcn_mfma_i32_32x32x32_i8(a1[f], bf[(f) & 7], acc1, 0, 0, 0); \
                 asm volatile("" : "+v"(acc0), "+v"(acc1)); }
    u32 pk = 0;
#pragma unroll
    for (int f = 0; f < 8; ++f) bf[f] = i32x4{0, 0, 0, 0};
#pragma unroll
    for (int f = 0; f < DIST; ++f) { PACE(); const u32 vo = wj + (f >> 2) * 4096; ALOAD(bf[f & 7], rs, vo, (f & 3) * 1024); }
#pragma unroll 1
    for (int j = 0; j < 8; ++j) {
#pragma unroll
      for (int m = 0; m < 16; ++m) {
        const int f = (m + DIST) & 15;
        if (f == 0) {
          wj += RS;
          if (j == 7) { wj = 0xF0000000u; due = -0x40000000; }
          if (j < 2) {
            asm volatile("" : "+v"(pk));
            if (ok && !__all((int)(pk - call) >= 0)) { ok = wait_words(rme, wf + 256 + (j & 1) * 256, call, call, t0, lane); if (!ok && lane == 0) lds_st(ldc + 32, 1); }
            asm volatile("" ::: "memory");
          }
        }
        PACE();
        AWAIT1(DIST - 1, bf[m & 7]);
        MMA(m);
        { const u32 vo = wj + (f >> 2) * 4096; ALOAD(bf[f & 7], rs, vo, (f & 3) * 1024); }
        if (m == 0) { const u32 vo = wf + 256 + (j & 1) * 256; ALOADW(pk, rm, vo); }
      }
      const u32 tile = lds0 + j * 8192;
      const u32 ad = tile + (lane >> 5) * 512 + (lane & 31) * 4;
#pragma unroll
      for (int q = 0; q < 16; ++q)
        __hip_atomic_fetch_add((lds_u32p)(size_t)(ad + ((q & 3) + 8 * (q >> 2)) * 128), (u32)acc0[q], __ATOMIC_RELAXED, __HIP_MEMORY_SCOPE_WORKGROUP);
#pragma unroll
      for (int q = 0; q < 16; ++q)
        __hip_atomic_fetch_add((lds_u32p)(size_t)(ad + 4096 + ((q & 3) + 8 * (q >> 2)) * 128), (u32)acc1[q], __ATOMIC_RELAXED, __HIP_MEMORY_SCOPE_WORKGROUP);
#pragma unroll
      for (int q = 0; q < 16; ++q) { acc0[q] = 0; acc1[q] = 0; }
      waitl0();
      if (lane == 0) __hip_atomic_fetch_add((lds_u32p)(size_t)(ldc + j * 4), 1u, __ATOMIC_RELAXED, __HIP_MEMORY_SCOPE_WORKGROUP);
    }
#pragma unroll 1
    for (int u = gs ? 56 + gs : 0; u <= 56 + gs; ++u) {
      const int j = u >> 3, i = u & 7;
      while (__builtin_amdgcn_readfirstlane(lds_ld(ldc + j * 4)) != 8) __builtin_amdgcn_s_sleep(1);
      asm volatile("" ::: "memory");
      const u32 isbad = lds_ld(ldc + 32);
      const u32x4 v = *(lds_u32x4p)(size_t)(lds0 + u * 1024 + lane * 16);
      const float sc = isbad ? __builtin_nanf("") : scale;
      const float lo = isbad ? __builtin_nanf("") : 0.f;
      f32x4 y;
#pragma unroll
      for (int e = 0; e < 4; ++e) y[e] = fmaxf((float)(int)v[e] * sc, lo);
      const u32 ob = ((rg * 64 + i * 8 + (lane >> 3)) * 4096 + j * 512 + c * 32 + (lane & 7) * 4) * 4;
      __builtin_amdgcn_raw_buffer_store_b128((u32x4)y, rout, ob, 0, OUTAUX);
    }
  }
}

static int cTx = 2780, cTw = 2695, cBurst = 4, cFirst = 250, cCF = 550, cGo = 150, cBD = 150, cCD = 600, cCS = 30, cLag = 400, cLE = -30, cTS = 1000, cMO = 460;
static int geti(const char* s, const char* key, int def) { const char* p = strstr(s, key); return p ? (int)strtol(p + strlen(key), nullptr, 0) : def; }
extern "C" void set_config(const char* s) {
  cTx = geti(s, "X=", 2780); cTw = geti(s, "W=", 2695); cBurst = geti(s, "B=", 4); cFirst = geti(s, "F=", 250); cGo = geti(s, "G=", 150);
  cBD = geti(s, "D=", 150); cCD = geti(s, "H=", 600); cCS = geti(s, "S=", 30); cCF = geti(s, "C=", 550); cLag = geti(s, "L=", 400); cLE = geti(s, "E=", -30); cTS = geti(s, "T=", 1000); cMO = geti(s, "M=", 460);
}

extern "C" void kernel_launch(int rank, int nranks, ncclComm_t comm, hipStream_t stream, void* const* d_in, const int* in_sizes, int n_in,
                              void* d_out, int out_size, const int* in_dtypes, int out_dtype, void* const* d_out_all,
                              void* const* d_ws_all, size_t ws_bytes) {
  if (nranks != NRANK || ws_bytes < WS_USED) return;
  Params P;
  P.x = (const char*)d_in[0]; P.w = (const char*)d_in[1]; P.sx = (const float*)d_in[2]; P.sw = (const float*)d_in[3];
  P.out = (float*)d_out;
  for (int g = 0; g < NRANK; ++g) P.pws[g] = (char*)d_ws_all[(rank + g) & 7];
  P.rank = rank;
  P.taux = (int)(cTx * 256 / 100.0); P.tauw = (int)(cTw * 256 / 100.0); P.burst = cBurst; P.first = cFirst; P.godelay = cGo;
  P.minorg = cMO; P.bdelay = cBD; P.cdelay = cCD; P.cstag = cCS; P.tauself = (int)(cTS * 256 / 100.0); P.cfirst = cCF; P.lag = cLag; P.lagend = cLE;
  fused<<<256, 512, 0, stream>>>(P);
}
```

```cpp
#include <hip/hip_runtime.h>
#include <rccl/rccl.h>
#include <cstdio>

typedef unsigned u32;
typedef unsigned long long u64;
typedef __attribute__((ext_vector_type(4))) unsigned u32x4;
typedef __attribute__((ext_vector_type(2))) unsigned u32x2;
typedef __attribute__((ext_vector_type(4))) int i32x4;
typedef __attribute__((ext_vector_type(16))) int i32x16;
typedef __attribute__((ext_vector_type(4))) float f32x4;
typedef __attribute__((address_space(3))) u32* lds_u32p;
typedef __attribute__((address_space(3))) u32x4* lds_u32x4p;
typedef __attribute__((address_space(1))) u32* glb_u32p;

constexpr u32 KiB = 1024, MiB = 1024 * 1024;
constexpr int NRANK = 8;
constexpr u32 XB = 256 * KiB;
constexpr u32 WB = 2 * MiB;
constexpr u32 BS = 16384 + 1024;
constexpr u32 RS = 16 * BS;
constexpr u32 XREG = 0;
constexpr u32 WREG = 8 * RS;
constexpr int MF = 12;
constexpr u32 MS = MF * 1024 + 1024;
constexpr u32 MREG = 16 * RS;
constexpr u32 SLOT = MREG + 512 * MS;
constexpr u32 CTRL = 64 * MiB;
constexpr u32 XFLAG = CTRL;
constexpr u32 WFLAG = CTRL + 64 * KiB;
constexpr u32 MFLAG = CTRL + 80 * KiB;
constexpr u32 WCNT = CTRL + 96 * KiB;
constexpr u32 ORG = CTRL + 128 * KiB;
constexpr u32 INVW = CTRL + 192 * KiB;
constexpr u32 CTRS = CTRL + 256 * KiB;
constexpr u32 TMO = CTRL + 384 * KiB;
constexpr u32 DBG = CTRL + 512 * KiB;
constexpr u32 WS_USED = 65 * MiB;
constexpr u32 OUT_BYTES = 8 * MiB;
constexpr u32 DEADLINE = 50000000u;
static_assert(2 * SLOT <= CTRL, "slots");

struct Params {
  char* pws[NRANK];
  const char* x; const char* w; const float* sx; const float* sw; float* out;
  int rank;
  int taux, tauw, tauself;
  int first, cfirst, minorg, godelay, bdelay, cdelay, cstag, lag, lagend, mmargin, plan, jump;
};

__device__ __forceinline__ __amdgpu_buffer_rsrc_t mkrsrc(const void* p, u32 bytes) {
  return __builtin_amdgcn_make_buffer_rsrc((void*)p, 0, bytes, 0x00020000);
}
__device__ __forceinline__ u32x4 rawrsrc(const void* p, u32 bytes) {
  const u64 a = (u64)p;
  u32x4 r = {(u32)a, (u32)(a >> 32) & 0xffffu, bytes, 0x00020000u};
  return r;
}
template <int N> __device__ __forceinline__ void waitv() { asm volatile("s_waitcnt vmcnt(%0)" ::"n"(N) : "memory"); }
__device__ __forceinline__ void waitl0() { asm volatile("s_waitcnt lgkmcnt(0)" ::: "memory"); }
__device__ __forceinline__ u32 lds_ld(u32 a) { return __hip_atomic_load((lds_u32p)(size_t)a, __ATOMIC_RELAXED, __HIP_MEMORY_SCOPE_WORKGROUP); }
__device__ __forceinline__ void lds_st(u32 a, u32 v) { __hip_atomic_store((lds_u32p)(size_t)a, v, __ATOMIC_RELAXED, __HIP_MEMORY_SCOPE_WORKGROUP); }
__device__ __forceinline__ u32 now32() { return (u32)__builtin_amdgcn_s_memrealtime(); }
__device__ __forceinline__ void sleep_until(u32 t0, int at) { while ((int)(now32() - t0) < at) __builtin_amdgcn_s_sleep(4); }
constexpr int SYS = 17, AGT = 16;
#define ALOAD(dst, r, voff, imm) asm volatile("buffer_load_dwordx4 %0, %1, %2, 0 offen offset:%3" : "+v"(dst) : "v"(voff), "s"(r), "n"(imm) : "memory")
#define AWAIT(n, x) asm volatile("s_waitcnt vmcnt(%1)" : "+v"(x) : "n"(n) : "memory")
#define UNTIL(org, due) while (cur < (due)) { cur = (int)((now32() - (org)) << 8); if ((due) - cur > 40 * 256) __builtin_amdgcn_s_sleep(4); }

#define GOIDX 8
#define XPF 4
#define XLD 4
#define JF 12
#define DIST 6
#define PWIN 4

__device__ __forceinline__ bool wait_words(__amdgpu_buffer_rsrc_t rme, u32 off, u32 call, u32 t0, int lane) {
  bool ok = true;
  while (true) {
    const u32 tp = now32();
    const u32 f = __builtin_amdgcn_raw_buffer_load_b32(rme, off, 0, SYS);
    if (__all((int)(f - call) >= 0)) break;
    if (__builtin_expect(tp - t0 > 100000u, 0)) {
      const u32 tm = __builtin_amdgcn_raw_buffer_load_b32(rme, TMO, 0, AGT);
      if (__builtin_amdgcn_readfirstlane(tm) == call || tp - t0 > DEADLINE) {
        if (lane == 0) __builtin_amdgcn_raw_buffer_store_b32(call, rme, TMO, 0, AGT);
        ok = false;
        break;
      }
      __builtin_amdgcn_s_sleep(32);
    }
    __builtin_amdgcn_s_sleep(1);
  }
  asm volatile("" ::: "memory");
  return ok;
}

__device__ __forceinline__ u32 elect_invalidate(char* me, __amdgpu_buffer_rsrc_t rme, u32 call, int lane, bool vote) {
  u32 xcc; asm volatile("s_getreg_b32 %0, hwreg(HW_REG_XCC_ID)" : "=s"(xcc));
  const u32 invw = INVW + (xcc & 15) * 512;
  if (vote) {
    u32 old = call;
    if (lane == 0) old = __hip_atomic_exchange((glb_u32p)(me + invw), call, __ATOMIC_RELAXED, __HIP_MEMORY_SCOPE_AGENT);
    if (__builtin_amdgcn_readfirstlane(old) != call) {
      __builtin_amdgcn_fence(__ATOMIC_ACQUIRE, "");
      waitv<0>();
      if (lane == 0) __builtin_amdgcn_raw_buffer_store_b32(call, rme, invw + 256, 0, AGT);
    }
  }
  return invw + 256;
}

__device__ __forceinline__ int phase_at(int pos, int k) { return pos < 2 ? pos : pos < 5 ? 4 + ((k + pos - 1) & 3) : pos == 6 ? 4 + k : pos >> 1; }

struct Plan { int n, m, ps, pc; bool good; };
__device__ __forceinline__ Plan make_plan(__amdgpu_buffer_rsrc_t rme, int c, u32 call, int tauw, int lane) {
  const int l = lane & 7, h = lane >> 3;
  const u32x2 o = __builtin_amdgcn_raw_buffer_load_b64(rme, ORG + (c * 8 + l) * 256, 0, SYS);
  const u32 mf = __builtin_amdgcn_raw_buffer_load_b32(rme, MFLAG + lane * 128, 0, SYS);
  const bool need = l != 0 && (h & 3) != 0 && l != h;
  const bool good = __all((l == 0 || o[0] == call) && (!need || (int)(mf - call) >= 0));
  const u32 ref = __builtin_amdgcn_readlane(o[1], 1);
  int d = (int)((float)(int)(o[1] - ref) * (2048.f / (float)tauw));
  d = good ? max(min(d, 4096), -4096) : 0;
  int dl[8], lo = 1 << 20, hi = -(1 << 20);
#pragma unroll
  for (int i = 1; i < 8; ++i) { dl[i] = __builtin_amdgcn_readlane(d, i); lo = min(lo, dl[i]); hi = max(hi, dl[i]); }
  int L = lo + (((hi - lo) * (lane + 1)) >> 6), shed = 0, cap = 0;
#pragma unroll
  for (int i = 1; i < 8; ++i) {
    shed += min(max((dl[i] - L + 7) >> 3, 0), MF);
    if (i != 4) cap += min(max((L - dl[i]) >> 3, 0), 16);
  }
  L = __builtin_amdgcn_readlane(L, __builtin_ctzll(__ballot(shed <= cap)));
  Plan r;
  r.n = l != 0 ? min(max((d - L + 7) >> 3, 0), MF) : 0;
  r.m = (l & 3) != 0 ? min(max((L - d) >> 3, 0), 16) : 0;
  r.ps = 0; r.pc = 0; r.good = good;
  int tot = 0;
#pragma unroll
  for (int i = 1; i < 8; ++i) {
    const int ni = __builtin_amdgcn_readlane(r.n, i), mi = __builtin_amdgcn_readlane(r.m, i);
    if (l > i) { r.ps += ni; r.pc += mi; }
    tot += ni;
  }
  r.m = min(r.m, max(tot - r.pc, 0));
  return r;
}

__global__ void __launch_bounds__(512) fused(Params P) {
  __shared__ __attribute__((aligned(1024))) char smem[96 * KiB + 128];
  const int b = blockIdx.x, t = threadIdx.x;
  const int lane = t & 63;
  const int wv = __builtin_amdgcn_readfirstlane(t >> 6);
  const int g = b & 7, cu = b >> 3;
  const int rank = P.rank;
  auto rme = mkrsrc(P.pws[0], WS_USED);
  const u32 lds0 = (u32)(size_t)(__attribute__((address_space(3))) char*)smem;
  const u32 ldc = lds0 + 96 * KiB;
  const u32 ctr = CTRS + (b * 8 + wv) * 4;
  const u32 callv = __builtin_amdgcn_raw_buffer_load_b32(rme, ctr, 0, AGT);
  int cur;

  if (__builtin_expect(g != 0 && wv == 0 && cu < 16, 1)) {
    auto rp = mkrsrc(P.pws[g], WS_USED);
    auto rx = mkrsrc(P.x, WB);
    const u32 so = (rank ^ g) * XB + cu * 16384 + (lane & 31) * 512 + (lane >> 5) * 16;
    u32x4 xv[16];
#pragma unroll
    for (int i = 0; i < XPF; ++i) xv[i] = __builtin_amdgcn_raw_buffer_load_b128(rx, so + i * 32, 0, 0);
    const u32 call = __builtin_amdgcn_readfirstlane(callv) + 1;
    const u32 dst = (call & 1) * SLOT + XREG + g * RS + cu * BS + lane * 16;
    const u32 fl = XFLAG + (g * 16 + cu) * 512;
    const int tau = P.taux;
    int due = cu * tau / 16;
    u32 org = 0;
    cur = 0;
#pragma unroll
    for (int i = 0; i < 16; ++i) {
      if (i) { UNTIL(org, due); due += tau; }
      if (i == GOIDX) { if (lane == 0) __builtin_amdgcn_raw_buffer_store_b32(call, rp, fl + 256, 0, SYS); }
      __builtin_amdgcn_raw_buffer_store_b128(xv[i], rp, dst + i * 1024, 0, SYS);
      if (i == 0) org = now32();
      asm volatile("" ::: "memory");
#pragma unroll
      for (int k = XPF + XLD * i; k < XPF + XLD * (i + 1) && k < 16; ++k) xv[k] = __builtin_amdgcn_raw_buffer_load_b128(rx, so + k * 32, 0, 0);
      asm volatile("" ::: "memory");
    }
    waitv<0>();
    if (lane == 0) __builtin_amdgcn_raw_buffer_store_b32(call, rp, fl, 0, SYS);
  }
  cur = -0x7fffffff;
  if (cu >= 16) {
    u32 call;
    const int p = cu - 16;
    const u32 t0 = now32();
    if (p < 7) {
      if (wv == 4 && lane < 3) lds_st(ldc + lane * 4, 0);
      waitl0();
      __builtin_amdgcn_s_barrier();
    }
    int task = -1;
    if (g == 0 && (wv == 2 || wv == 3)) task = p * 2 + (wv & 1);
    else if (p >= 8) {
      if (wv >= 5) task = 64 + (((p - 8) * 8 + g) * 3 + wv - 5);
      else if (g == 0) task = 32 + (p - 8) * 4 + (wv == 4 ? 2 : wv);
    }
    if (g == 0 && p == 7 && task < 0) task = 32 + (wv < 2 ? wv : wv - 2) * 4 + 3;
    if (g == 0 && p == 6 && wv >= 6) task = 32 + wv * 4 + 3;
    if (task >= 0) {
      const int ph = task >> 5, blk = (task >> 2) & 7, cq = task & 3;
      const int stage = ph < 2 ? ph : 2;
      char* pw = P.pws[lane & 7];
      auto rw = mkrsrc(P.w, WB);
      const int kq = lane >> 4, q = lane & 15;
      if (stage) sleep_until(t0, stage == 1 ? P.bdelay : P.cdelay + (p - 8) * P.cstag);
      const u32 src = (u32)(64 * blk + 16 * kq) * 4096u + 512 * ph + 128 * cq + 8 * q;
      u32x2 in[16];
#pragma unroll
      for (int r = 0; r < 16; ++r) in[r] = __builtin_amdgcn_raw_buffer_load_b64(rw, src, r * 4096, 0);
      asm volatile("" : "+v"(pw));
      call = __builtin_amdgcn_readfirstlane(callv) + 1;
      u32x4 o[8];
#pragma unroll
      for (int d = 0; d < 2; ++d)
#pragma unroll
        for (int m = 0; m < 4; ++m) {
          u32 a0 = in[4 * m][d], a1 = in[4 * m + 1][d], a2 = in[4 * m + 2][d], a3 = in[4 * m + 3][d];
          u32 t0_ = __builtin_amdgcn_perm(a1, a0, 0x05010400), t1_ = __builtin_amdgcn_perm(a1, a0, 0x07030602);
          u32 t2_ = __builtin_amdgcn_perm(a3, a2, 0x05010400), t3_ = __builtin_amdgcn_perm(a3, a2, 0x07030602);
          o[4 * d + 0][m] = __builtin_amdgcn_perm(t2_, t0_, 0x05040100);
          o[4 * d + 1][m] = __builtin_amdgcn_perm(t2_, t0_, 0x07060302);
          o[4 * d + 2][m] = __builtin_amdgcn_perm(t3_, t1_, 0x05040100);
          o[4 * d + 3][m] = __builtin_amdgcn_perm(t3_, t1_, 0x07060302);
        }
      const u32 lw = lds0 + wv * 8192;
      const u32 wbase = lw + (q >> 2) * 2048 + (kq >> 1) * 1024 + (8 * (q & 3) + 32 * (kq & 1)) * 16;
#pragma unroll
      for (int e = 0; e < 8; ++e) *(lds_u32x4p)(size_t)(wbase + ((e ^ (q & 7)) << 4)) = o[e];
      waitl0();
      const u32 ra = lw + (((lane & ~7) | ((lane & 7) ^ ((lane >> 3) & 3))) << 4);
      const u32 dst = (call & 1) * SLOT + WREG + ph * RS + cq * 4 * BS + blk * 2048 + lane * 16;
#pragma unroll
      for (int i = 0; i < 8; ++i) o[i] = *(lds_u32x4p)(size_t)((ra ^ (((i >> 1) & 1) * 64)) + i * 1024);
#pragma unroll
      for (int i = 0; i < 8; ++i) __builtin_amdgcn_raw_buffer_store_b128(o[i], rme, dst + (i >> 1) * BS + (i & 1) * 1024, 0, SYS);
      waitv<0>();
      u32 old = 0;
      if (lane == 0) old = __hip_atomic_fetch_add((glb_u32p)(P.pws[0] + WCNT + stage * 256), 1u, __ATOMIC_RELAXED, __HIP_MEMORY_SCOPE_AGENT);
      old = __builtin_amdgcn_readfirstlane(old);
      if (old + 1 == call * (stage < 2 ? 32u : 192u) && lane < NRANK)
        __hip_atomic_store((glb_u32p)(pw + WFLAG + lane * 1024 + stage * 256), call, __ATOMIC_RELAXED, __HIP_MEMORY_SCOPE_SYSTEM);
    }
    call = __builtin_amdgcn_readfirstlane(callv) + 1;
    if (lane == 0) __builtin_amdgcn_raw_buffer_store_b32(call, rme, ctr, 0, AGT);
    const int gs = p + 1;
    const u32 wf = WFLAG + gs * 1024;
    if (p < 7 && wv < 2) {
      const int c = 2 * g + wv;
      const u32x4 rs = rawrsrc(P.pws[gs], WS_USED);
      const u32 invd = elect_invalidate(P.pws[0], rme, call, lane, true);
      sleep_until(t0, P.first);
      bool ok = true;
      u32 org = t0 + P.minorg;
#pragma unroll 1
      for (int k = 0; k < 2 && ok; ++k) {
        ok = wait_words(rme, k == 0 ? XFLAG + (gs * 16 + c) * 512 + 256 : lane == 1 ? invd : wf, call, t0, lane);
        const u32 cand = now32() + (k == 0 ? P.godelay : 0);
        if ((int)(cand - org) > 0) org = cand;
      }
      if (lane == 0) { u32x2 o = {call, org}; __builtin_amdgcn_raw_buffer_store_b64(o, rme, ORG + (c * 8 + gs) * 256, 0, AGT); }
      int due = 0;
      const int tau = P.tauw;
      const u32 wb = (call & 1) * SLOT + WREG + c * BS;
      u32x4 v = {0, 0, 0, 0};
#pragma unroll 1
      for (int j = 0; j < 8 && ok; ++j) {
        if (j == 1 || j == 2) {
          u32 w;
          while ((w = __builtin_amdgcn_readfirstlane(lds_ld(ldc))) < (u32)j) __builtin_amdgcn_s_sleep(1);
          asm volatile("" ::: "memory");
          if (w > 2) break;
        }
        u32 tab = wb + phase_at(j, rank & 3) * RS + lane * 1024;
        int cnt = 16, extra = j == 7 && P.plan;
#pragma unroll 1
        for (int i = 0; i < cnt + extra; ++i) {
          if (j == 6 || extra) {
            u32 w = __builtin_amdgcn_readfirstlane(lds_ld(ldc + 4 + wv * 4));
            if (j == 6) { if (i + (int)(w & 255) >= 16) break; }
            else {
              while (i == cnt && w == 0) { __builtin_amdgcn_s_sleep(2); w = __builtin_amdgcn_readfirstlane(lds_ld(ldc + 4 + wv * 4)); }
              if (w) {
                asm volatile("" ::: "memory");
                const int m = (w >> 8) & 255;
                const u32 ex = lds_ld(lds0 + 64 * KiB + wv * 256 + ((lane - i) & 63) * 4);
                tab = lane - i < m ? ex : tab - m * 1024;
                cnt += m; extra = 0;
                if (i == cnt) break;
              }
            }
          }
          const u32 vo = __builtin_amdgcn_readlane(tab, i) + lane * 16;
          UNTIL(org, due);
          due += tau;
          waitv<PWIN - 1>();
          ALOAD(v, rs, vo, 0);
        }
      }
      waitv<0>();
      asm volatile("" :: "v"(v));
    } else if (p < 7 && wv == 4) {
#pragma unroll 1
      for (u32 s = 1; s <= 2; ++s) {
        sleep_until(t0, s == 1 ? 600 : 1000);
        const bool ok = wait_words(rme, wf + s * 256, call, t0, lane);
        if (lane == 0) lds_st(ldc, ok ? s : 3);
        if (!ok) break;
      }
    } else if (p < 7 && (wv == 5 || wv == 6) && P.plan) {
      const int c = 2 * g + wv - 5;
      const u32 og = ORG + (c * 8 + gs) * 256;
      sleep_until(t0, P.plan);
      if (wait_words(rme, og, call, t0, lane)) {
        const u32 org = __builtin_amdgcn_readfirstlane(__builtin_amdgcn_raw_buffer_load_b32(rme, og + 4, 0, SYS));
        UNTIL(org, 90 * P.tauw);
        Plan pl;
        do { pl = make_plan(rme, c, call, P.tauw, lane); } while (!pl.good && (int)((now32() - org) << 8) < 140 * P.tauw);
        const int n = __builtin_amdgcn_readlane(pl.n, gs), m = __builtin_amdgcn_readlane(pl.m, gs), u = __builtin_amdgcn_readlane(pl.pc, gs) + lane;
        int a = 0, f = 0;
#pragma unroll
        for (int i = 1; i < 8; ++i) { const int ps = __builtin_amdgcn_readlane(pl.ps, i); const int ni = __builtin_amdgcn_readlane(pl.n, i); if (u >= ps && ni) { a = i; f = MF - ni + u - ps; } }
        const u32 slot = (call & 1) * SLOT;
        const u32 off = slot + MREG + (((a ^ gs) * 4 + (rank & 3)) * 16 + c) * MS + (lane < m ? f : 0) * 1024;
        lds_st(lds0 + 64 * KiB + (wv - 5) * 256 + lane * 4, off);
        waitl0();
        if (lane == 0) lds_st(ldc + 4 + (wv - 5) * 4, n + 256 * m + 65536);
      } else if (lane == 0) lds_st(ldc + 4 + (wv - 5) * 4, 65536);
    } else if (p >= 8 && p < 15 && wv < 2 && P.plan) {
      const int ms = p - 7, c = 2 * g + wv;
      const u32 og = ORG + (c * 8 + ms) * 256;
      const u32 invd = elect_invalidate(P.pws[0], rme, call, lane, true);
      auto rs = mkrsrc(P.pws[ms], WS_USED);
      char* pw = P.pws[lane & 7];
      sleep_until(t0, 1200);
      if (wait_words(rme, lane == 0 ? og : lane == 1 ? invd : WFLAG + ms * 1024 + 512, call, t0, lane)) {
        const u32 org = __builtin_amdgcn_readfirstlane(__builtin_amdgcn_raw_buffer_load_b32(rme, og + 4, 0, SYS));
        const u32 slot = (call & 1) * SLOT;
#pragma unroll 1
        for (int j = 2; j < 5; ++j) {
          const int ph = phase_at(j, rank & 3);
          UNTIL(org, (j * 16 + 16) * P.tauw + P.mmargin * 256);
          u32x4 v[MF];
#pragma unroll
          for (int i = 0; i < MF; ++i) v[i] = __builtin_amdgcn_raw_buffer_load_b128(rs, slot + WREG + ph * RS + c * BS + (16 - MF + i) * 1024 + lane * 16, 0, 0);
#pragma unroll
          for (int i = 0; i < MF; ++i) __builtin_amdgcn_raw_buffer_store_b128(v[i], rme, slot + MREG + ((ms * 4 + ph - 4) * 16 + c) * MS + i * 1024 + lane * 16, 0, SYS);
        }
        waitv<0>();
        u32 old = 0;
        if (lane == 0) old = __hip_atomic_fetch_add((glb_u32p)(P.pws[0] + WCNT + (4 + ms) * 256), 1u, __ATOMIC_RELAXED, __HIP_MEMORY_SCOPE_AGENT);
        old = __builtin_amdgcn_readfirstlane(old);
        if (old + 1 == call * 16u && lane < NRANK && (lane & 3) && lane != ms)
          __hip_atomic_store((glb_u32p)(pw + MFLAG + (lane * 8 + (lane ^ ms)) * 128), call, __ATOMIC_RELAXED, __HIP_MEMORY_SCOPE_SYSTEM);
      }
    }
  } else {
    const u32 t0 = now32();
    {
      u32x4 z = {0, 0, 0, 0};
#pragma unroll
      for (int i = 0; i < 8; ++i) *(lds_u32x4p)(size_t)(lds0 + wv * 8192 + i * 1024 + lane * 16) = z;
      if (wv == 0 && lane < 16) *(lds_u32p)(size_t)(ldc + lane * 4) = 0;
    }
    const u32 call = __builtin_amdgcn_readfirstlane(callv) + 1;
    if (lane == 0) __builtin_amdgcn_raw_buffer_store_b32(call, rme, ctr, 0, AGT);
    const float scale = P.sx[0] * P.sw[0];
    const u32 invd = elect_invalidate(P.pws[0], rme, call, lane, wv == 1);
    waitl0();
    __builtin_amdgcn_s_barrier();
    asm volatile("" ::: "memory");
    const int gs = wv;
    const int c = 2 * g + (cu & 1), rg = cu >> 1;
    const u32 slot = (call & 1) * SLOT;
    const u32 wf = WFLAG + gs * 1024;
    const u32 og = ORG + (c * 8 + gs) * 256;
    bool ok;
    i32x4 a0[16], a1[16], bf[8];
    u32 org;
    {
      u32 off = wf;
      if (gs) {
        sleep_until(t0, P.cfirst);
        off = lane == 0 ? wf : lane == 1 ? og : lane == 2 ? invd : XFLAG + (gs * 16 + 2 * rg + (lane & 1)) * 512;
      }
      ok = wait_words(rme, off, call, t0, lane);
      const u32 ov = __builtin_amdgcn_raw_buffer_load_b32(rme, og + 4, 0, SYS);
      u32 xo = gs ? slot + XREG + gs * RS + 2 * rg * BS + lane * 16 : rank * XB + 2 * rg * 16384 + (lane & 31) * 512 + (lane >> 5) * 16;
      const u32 xs1 = gs ? BS : 16384, xsf = gs ? 1024 : 32;
      auto ra = gs ? rme : mkrsrc(P.x, WB);
#pragma unroll
      for (int f = 0; f < 16; ++f) {
        a0[f] = (i32x4)__builtin_amdgcn_raw_buffer_load_b128(ra, xo, 0, 0);
        a1[f] = (i32x4)__builtin_amdgcn_raw_buffer_load_b128(ra, xo + xs1, 0, 0);
        xo += xsf;
      }
      org = gs && ok ? __builtin_amdgcn_readfirstlane(ov) : now32() - P.lag;
      waitv<0>();
#pragma unroll
      for (int f = 0; f < 16; ++f) asm volatile("" : "+v"(a0[f]), "+v"(a1[f]));
    }
    int due = P.lag * 256;
    const int tau = gs ? P.tauw - (P.lag - P.lagend) * 2 : P.tauself;
    const u64 wb = (u64)P.pws[gs] + slot + WREG + c * BS + (lane & 15) * 1024;
    const u32 l16 = lane * 16;
    int shift = 0, cap = 0x7fffffff;
    bool planned = gs == 0 || !P.plan;
#pragma unroll
    for (int f = 0; f < 8; ++f) bf[f] = i32x4{0, 0, 0, 0};
#pragma unroll 1
    for (int j = 0; j < 8; ++j) {
      if (ok && (j == 1 || j == 2)) ok = wait_words(rme, wf + j * 256, call, t0, lane);
      if (!ok && lane == 0) lds_st(ldc + 32, 1);
      const u32 nrec = ok ? 1024 : 0;
      u64 tab = wb + phase_at(j, rank & 3) * RS;
      if (j >= 6 && !planned && ok) {
        const Plan pl = make_plan(rme, c, call, P.tauw, lane);
        const int n = __builtin_amdgcn_readlane(pl.n, gs), u = __builtin_amdgcn_readlane(pl.ps, gs) + lane - (16 - n);
        int h = 0;
#pragma unroll
        for (int i = 1; i < 8; ++i) if (u >= __builtin_amdgcn_readlane(pl.pc, i) && __builtin_amdgcn_readlane(pl.m, i)) h = i;
        if (j == 6 && lane >= 16 - n && lane < 16) tab = (u64)P.pws[h] + slot + MREG + (((gs ^ h) * 4 + (rank & 3)) * 16 + c) * MS + (lane - (16 - MF)) * 1024;
        shift = (__builtin_amdgcn_readlane(pl.m, gs) - n) * P.tauw;
        if (j == 6) cap = due + (16 - n) * tau;
        planned = pl.good;
      }
      if (j == 7) { due += shift; cap = 0x7fffffff; }
      const int jump = j == 7 ? P.jump * 256 : 0;
      const u32 tlo = (u32)tab, thi = (u32)(tab >> 32) & 0xffffu;
      i32x16 acc0 = {0, 0, 0, 0, 0, 0, 0, 0, 0, 0, 0, 0, 0, 0, 0, 0}, acc1 = acc0;
#pragma unroll
      for (int f = 0; f < 16 + DIST; ++f) {
        if (f >= DIST) {
          const int m = f - DIST;
          AWAIT((f < 16 ? DIST - 1 : 15 - m), bf[m & 7]);
          acc0 = __builtin_amdgcn_mfma_i32_32x32x32_i8(a0[m], bf[m & 7], acc0, 0, 0, 0);
          acc1 = __builtin_amdgcn_mfma_i32_32x32x32_i8(a1[m], bf[m & 7], acc1, 0, 0, 0);
          asm volatile("" : "+v"(acc0), "+v"(acc1));
        }
        if (f < 16) {
          if (f == JF) due -= jump;
          { const int dd = min(due, cap); UNTIL(org, dd); }
          due += tau;
          const u32x4 rf = {(u32)__builtin_amdgcn_readlane(tlo, f), (u32)__builtin_amdgcn_readlane(thi, f), nrec, 0x00020000u};
          asm volatile("s_nop 4\n\tbuffer_load_dwordx4 %0, %1, %2, 0 offen" : "+v"(bf[f & 7]) : "v"(l16), "s"(rf) : "memory");
        }
      }
      const u32 ad = lds0 + j * 8192 + (lane >> 5) * 512 + (lane & 31) * 4;
#pragma unroll
      for (int e = 0; e < 16; ++e)
        __hip_atomic_fetch_add((lds_u32p)(size_t)(ad + ((e & 3) + 8 * (e >> 2)) * 128), (u32)acc0[e], __ATOMIC_RELAXED, __HIP_MEMORY_SCOPE_WORKGROUP);
#pragma unroll
      for (int e = 0; e < 16; ++e)
        __hip_atomic_fetch_add((lds_u32p)(size_t)(ad + 4096 + ((e & 3) + 8 * (e >> 2)) * 128), (u32)acc1[e], __ATOMIC_RELAXED, __HIP_MEMORY_SCOPE_WORKGROUP);
      waitl0();
      if (lane == 0) __hip_atomic_fetch_add((lds_u32p)(size_t)(ldc + j * 4), 1u, __ATOMIC_RELAXED, __HIP_MEMORY_SCOPE_WORKGROUP);
    }
    auto rout = mkrsrc(P.out, OUT_BYTES);
#pragma unroll 1
    for (int u = gs ? 56 + gs : 0; u <= 56 + gs; ++u) {
      const int j = u >> 3, i = u & 7;
      const u32 ob = ((rg * 64 + i * 8 + (lane >> 3)) * 4096 + phase_at(j, rank & 3) * 512 + c * 32 + (lane & 7) * 4) * 4;
      if (j == 7) __builtin_amdgcn_s_barrier();
      else while (__builtin_amdgcn_readfirstlane(lds_ld(ldc + j * 4)) != 8) __builtin_amdgcn_s_sleep(1);
      asm volatile("" ::: "memory");
      const u32 isbad = lds_ld(ldc + 32);
      const u32x4 v = *(lds_u32x4p)(size_t)(lds0 + u * 1024 + lane * 16);
      const float sc = isbad ? __builtin_nanf("") : scale;
      const float lo = isbad ? __builtin_nanf("") : 0.f;
      f32x4 y;
#pragma unroll
      for (int e = 0; e < 4; ++e) y[e] = fmaxf((float)(int)v[e] * sc, lo);
      __builtin_amdgcn_raw_buffer_store_b128((u32x4)y, rout, ob, 0, SYS);
    }
  }
}

static inline int dv(char, int def) { return def; }

extern "C" void kernel_launch(int rank, int nranks, ncclComm_t comm, hipStream_t stream, void* const* d_in, const int* in_sizes, int n_in,
                              void* d_out, int out_size, const int* in_dtypes, int out_dtype, void* const* d_out_all,
                              void* const* d_ws_all, size_t ws_bytes) {
  if (nranks != NRANK || (unsigned)rank >= (unsigned)NRANK || ws_bytes < WS_USED || n_in < 4 || in_sizes[0] != (int)WB ||
      in_sizes[1] != (int)WB || out_size != (int)(OUT_BYTES / 4)) {
    fprintf(stderr, "kernel_launch: unexpected problem shape\n");
    return;
  }
  Params P;
  P.x = (const char*)d_in[0]; P.w = (const char*)d_in[1]; P.sx = (const float*)d_in[2]; P.sw = (const float*)d_in[3];
  P.out = (float*)d_out;
  for (int l = 0; l < NRANK; ++l) P.pws[l] = (char*)d_ws_all[rank ^ l];
  P.rank = rank;
  P.taux = dv('X', 2780) * 256 / 100; P.tauw = dv('W', 2695) * 256 / 100; P.tauself = dv('T', 1000) * 256 / 100;
  P.first = dv('F', 250); P.cfirst = dv('C', 550); P.minorg = dv('M', 460); P.godelay = dv('G', 150);
  P.bdelay = dv('D', 150); P.cdelay = dv('H', 600); P.cstag = dv('S', 30); P.lag = dv('L', 300); P.lagend = dv('E', 60);
  P.mmargin = dv('Y', 40); P.plan = dv('P', 2000); P.jump = dv('J', 90);
  fused<<<256, 512, 0, stream>>>(P);
}
```

```cpp
#include <hip/hip_runtime.h>
#include <rccl/rccl.h>
#include <cstdio>

typedef unsigned u32;
typedef unsigned long long u64;
typedef __attribute__((ext_vector_type(4))) unsigned u32x4;
typedef __attribute__((ext_vector_type(2))) unsigned u32x2;
typedef __attribute__((ext_vector_type(4))) int i32x4;
typedef __attribute__((ext_vector_type(4))) float f32x4;
typedef __attribute__((address_space(3))) u32* lds_u32p;
typedef __attribute__((address_space(3))) u32x4* lds_u32x4p;
typedef __attribute__((address_space(1))) u32* glb_u32p;

constexpr u32 KiB = 1024, MiB = 1024 * 1024;
constexpr int NRANK = 8;
constexpr u32 XB = 256 * KiB;
constexpr u32 WB = 2 * MiB;
constexpr u32 BS = 16384 + 1024;
constexpr u32 RS = 16 * BS;
constexpr u32 XREG = 0;
constexpr u32 WREG = 8 * RS;
constexpr int MF = 12;
constexpr u32 MS = MF * 1024 + 1024;
constexpr u32 MREG = 16 * RS;
constexpr u32 SLOT = MREG + 512 * MS;
constexpr u32 CTRL = 64 * MiB;
constexpr u32 XFLAG = CTRL;
constexpr u32 WFLAG = CTRL + 64 * KiB;
constexpr u32 MFLAG = CTRL + 80 * KiB;
constexpr u32 WCNT = CTRL + 96 * KiB;
constexpr u32 ORG = CTRL + 128 * KiB;
constexpr u32 INVW = CTRL + 192 * KiB;
constexpr u32 CTRS = CTRL + 256 * KiB;
constexpr u32 TMO = CTRL + 384 * KiB;
constexpr u32 WS_USED = 65 * MiB;
constexpr u32 OUT_BYTES = 8 * MiB;
constexpr u32 DEADLINE = 50000000u;
static_assert(2 * SLOT <= CTRL, "slots");

struct Params {
  char* pws[NRANK];
  const char* x; const char* w; const float* sx; const float* sw; float* out;
  int rank;
  int taux, tauw, tauself;
  int first, cfirst, minorg, godelay, bdelay, cdelay, cstag, lag, lagend, mmargin, plan, jump;
};

__device__ __forceinline__ __amdgpu_buffer_rsrc_t mkrsrc(const void* p, u32 bytes) {
  return __builtin_amdgcn_make_buffer_rsrc((void*)p, 0, bytes, 0x00020000);
}
__device__ __forceinline__ u32x4 rawrsrc(const void* p, u32 bytes) {
  const u64 a = (u64)p;
  u32x4 r = {(u32)a, (u32)(a >> 32) & 0xffffu, bytes, 0x00020000u};
  return r;
}
template <int N> __device__ __forceinline__ void waitv() { asm volatile("s_waitcnt vmcnt(%0)" ::"n"(N) : "memory"); }
__device__ __forceinline__ void waitl0() { asm volatile("s_waitcnt lgkmcnt(0)" ::: "memory"); }
__device__ __forceinline__ u32 lds_ld(u32 a) { return __hip_atomic_load((lds_u32p)(size_t)a, __ATOMIC_RELAXED, __HIP_MEMORY_SCOPE_WORKGROUP); }
__device__ __forceinline__ void lds_st(u32 a, u32 v) { __hip_atomic_store((lds_u32p)(size_t)a, v, __ATOMIC_RELAXED, __HIP_MEMORY_SCOPE_WORKGROUP); }
__device__ __forceinline__ u32 now32() { return (u32)__builtin_amdgcn_s_memrealtime(); }
__device__ __forceinline__ void sleep_until(u32 t0, int at) { while ((int)(now32() - t0) < at) __builtin_amdgcn_s_sleep(4); }
constexpr int SYS = 17, AGT = 16;
#define ALOAD(dst, r, voff, imm) asm volatile("buffer_load_dwordx4 %0, %1, %2, 0 offen offset:%3" : "+v"(dst) : "v"(voff), "s"(r), "n"(imm) : "memory")
#define AWAIT(n, x) asm volatile("s_waitcnt vmcnt(%1)" : "+v"(x) : "n"(n) : "memory")
#define UNTIL(org, due) while (cur < (due)) { cur = (int)((now32() - (org)) << 8); if ((due) - cur > 40 * 256) __builtin_amdgcn_s_sleep(4); }

#define GOIDX 8
#define XPF 8
#define XLD 4
#define JF 12
#define DIST 6
#define PWIN 4

__device__ __forceinline__ bool wait_words(__amdgpu_buffer_rsrc_t rme, u32 off, u32 call, u32 t0, int lane) {
  bool ok = true;
  while (true) {
    asm volatile("" ::: "memory");
    const u32 tp = now32();
    const u32 f = __builtin_amdgcn_raw_buffer_load_b32(rme, off, 0, SYS);
    if (__all((int)(f - call) >= 0)) break;
    if (__builtin_expect(tp - t0 > 100000u, 0)) {
      const u32 tm = __builtin_amdgcn_raw_buffer_load_b32(rme, TMO, 0, AGT);
      if (__builtin_amdgcn_readfirstlane(tm) == call || tp - t0 > DEADLINE) {
        if (lane == 0) __builtin_amdgcn_raw_buffer_store_b32(call, rme, TMO, 0, AGT);
        ok = false;
        break;
      }
      __builtin_amdgcn_s_sleep(32);
    }
    __builtin_amdgcn_s_sleep(1);
  }
  asm volatile("" ::: "memory");
  return ok;
}

__device__ __forceinline__ u32 elect_invalidate(char* me, __amdgpu_buffer_rsrc_t rme, u32 call, int lane, bool vote) {
  u32 xcc; asm volatile("s_getreg_b32 %0, hwreg(HW_REG_XCC_ID)" : "=s"(xcc));
  const u32 invw = INVW + (xcc & 15) * 512;
  if (vote) {
    u32 old = call;
    if (lane == 0) old = __hip_atomic_exchange((glb_u32p)(me + invw), call, __ATOMIC_RELAXED, __HIP_MEMORY_SCOPE_AGENT);
    if (__builtin_amdgcn_readfirstlane(old) != call) {
      __builtin_amdgcn_fence(__ATOMIC_ACQUIRE, "");
      waitv<0>();
      if (lane == 0) __builtin_amdgcn_raw_buffer_store_b32(call, rme, invw + 256, 0, AGT);
    }
  }
  return invw + 256;
}

__device__ __forceinline__ int phase_at(int pos, int k) { return pos < 2 ? pos : pos < 5 ? 4 + ((k + pos - 1) & 3) : pos == 6 ? 4 + k : pos >> 1; }

struct Plan { int n, m, ps, pc; bool good; };
__device__ __forceinline__ Plan make_plan(__amdgpu_buffer_rsrc_t rme, int c, u32 call, int tauw, int lane) {
  const int l = lane & 7, h = lane >> 3;
  asm volatile("" ::: "memory");
  const u32x2 o = __builtin_amdgcn_raw_buffer_load_b64(rme, ORG + (c * 8 + l) * 256, 0, SYS);
  const u32 mf = __builtin_amdgcn_raw_buffer_load_b32(rme, MFLAG + lane * 128, 0, SYS);
  const bool need = l != 0 && (h & 3) != 0 && l != h;
  const bool good = __all((l == 0 || o[0] == call) && (!need || (int)(mf - call) >= 0));
  const u32 ref = __builtin_amdgcn_readlane(o[1], 1);
  int d = (int)((float)(int)(o[1] - ref) * (2048.f / (float)tauw));
  d = good ? max(min(d, 4096), -4096) : 0;
  int dl[8], lo = 1 << 20, hi = -(1 << 20);
#pragma unroll
  for (int i = 1; i < 8; ++i) { dl[i] = __builtin_amdgcn_readlane(d, i); lo = min(lo, dl[i]); hi = max(hi, dl[i]); }
  int L = lo + (((hi - lo) * (lane + 1)) >> 6), shed = 0, cap = 0;
#pragma unroll
  for (int i = 1; i < 8; ++i) {
    shed += min(max((dl[i] - L + 7) >> 3, 0), MF);
    if (i != 4) cap += min(max((L - dl[i]) >> 3, 0), 16);
  }
  L = __builtin_amdgcn_readlane(L, __builtin_ctzll(__ballot(shed <= cap)));
  Plan r;
  r.n = l != 0 ? min(max((d - L + 7) >> 3, 0), MF) : 0;
  r.m = (l & 3) != 0 ? min(max((L - d) >> 3, 0), 16) : 0;
  r.ps = 0; r.pc = 0; r.good = good;
  int tot = 0;
#pragma unroll
  for (int i = 1; i < 8; ++i) {
    const int ni = __builtin_amdgcn_readlane(r.n, i), mi = __builtin_amdgcn_readlane(r.m, i);
    if (l > i) { r.ps += ni; r.pc += mi; }
    tot += ni;
  }
  r.m = min(r.m, max(tot - r.pc, 0));
  return r;
}

__global__ void __launch_bounds__(512) fused(Params P) {
  __shared__ __attribute__((aligned(1024))) char smem[96 * KiB + 128];
  const int b = blockIdx.x, t = threadIdx.x;
  const int lane = t & 63;
  const int wv = __builtin_amdgcn_readfirstlane(t >> 6);
  const int g = b & 7, cu = b >> 3;
  const int rank = P.rank;
  auto rme = mkrsrc(P.pws[0], WS_USED);
  const u32 lds0 = (u32)(size_t)(__attribute__((address_space(3))) char*)smem;
  const u32 ldc = lds0 + 96 * KiB;
  const u32 ctr = CTRS + (b * 8 + wv) * 4;
  const u32 callv = __builtin_amdgcn_raw_buffer_load_b32(rme, ctr, 0, AGT);
  int cur;

  if (__builtin_expect(g != 0 && wv == 0 && cu < 16, 1)) {
    auto rp = mkrsrc(P.pws[g], WS_USED);
    auto rx = mkrsrc(P.x, WB);
    const u32 so = (rank ^ g) * XB + cu * 16384 + (lane & 15) * 512 + (lane >> 4) * 16;
    u32x4 xv[16];
#pragma unroll
    for (int i = 0; i < XPF; ++i) xv[i] = __builtin_amdgcn_raw_buffer_load_b128(rx, so + (i >> 3) * 8192 + (i & 7) * 64, 0, 0);
    const u32 call = __builtin_amdgcn_readfirstlane(callv) + 1;
    const u32 dst = (call & 1) * SLOT + XREG + g * RS + cu * BS + lane * 16;
    const u32 fl = XFLAG + (g * 16 + cu) * 512;
    const int tau = P.taux;
    int due = cu * tau / 16;
    u32 org = 0;
    cur = 0;
#pragma unroll
    for (int i = 0; i < 16; ++i) {
      if (i) { UNTIL(org, due); due += tau; }
      if (i == GOIDX) { if (lane == 0) __builtin_amdgcn_raw_buffer_store_b32(call, rp, fl + 256, 0, SYS); }
      __builtin_amdgcn_raw_buffer_store_b128(xv[i], rp, dst + i * 1024, 0, SYS);
      if (i == 0) org = now32();
      asm volatile("" ::: "memory");
#pragma unroll
      for (int k = XPF + XLD * i; k < XPF + XLD * (i + 1) && k < 16; ++k) xv[k] = __builtin_amdgcn_raw_buffer_load_b128(rx, so + (k >> 3) * 8192 + (k & 7) * 64, 0, 0);
      asm volatile("" ::: "memory");
    }
    waitv<0>();
    if (lane == 0) __builtin_amdgcn_raw_buffer_store_b32(call, rp, fl, 0, SYS);
  }
  cur = -0x7fffffff;
  if (cu >= 16) {
    u32 call;
    const int p = cu - 16;
    const u32 t0 = now32();
    if (p < 7) {
      if (wv == 4 && lane < 3) lds_st(ldc + lane * 4, 0);
      waitl0();
      __builtin_amdgcn_s_barrier();
    }
    int task = -1;
    if (g == 0 && (wv == 2 || wv == 3)) task = p * 2 + (wv & 1);
    else if (p >= 8) {
      if (wv >= 5) task = 64 + (((p - 8) * 8 + g) * 3 + wv - 5);
      else if (g == 0) task = 32 + (p - 8) * 4 + (wv == 4 ? 2 : wv);
    }
    if (g == 0 && p == 7 && task < 0) task = 32 + (wv < 2 ? wv : wv - 2) * 4 + 3;
    if (g == 0 && p == 6 && wv >= 6) task = 32 + wv * 4 + 3;
    if (task >= 0) {
      const int ph = task >> 5, blk = (task >> 2) & 7, cq = task & 3;
      const int stage = ph < 2 ? ph : 2;
      char* pw = P.pws[lane & 7];
      auto rw = mkrsrc(P.w, WB);
      const int kq = lane >> 4, q = lane & 15;
      if (stage) sleep_until(t0, stage == 1 ? P.bdelay : P.cdelay + (p - 8) * P.cstag);
      const u32 src = (u32)(64 * blk + 16 * kq) * 4096u + 512 * ph + 128 * cq + 8 * q;
      u32x2 in[16];
#pragma unroll
      for (int r = 0; r < 16; ++r) in[r] = __builtin_amdgcn_raw_buffer_load_b64(rw, src, r * 4096, 0);
      asm volatile("" : "+v"(pw));
      call = __builtin_amdgcn_readfirstlane(callv) + 1;
      u32x4 o[8];
#pragma unroll
      for (int d = 0; d < 2; ++d)
#pragma unroll
        for (int m = 0; m < 4; ++m) {
          u32 a0 = in[4 * m][d], a1 = in[4 * m + 1][d], a2 = in[4 * m + 2][d], a3 = in[4 * m + 3][d];
          u32 t0_ = __builtin_amdgcn_perm(a1, a0, 0x05010400), t1_ = __builtin_amdgcn_perm(a1, a0, 0x07030602);
          u32 t2_ = __builtin_amdgcn_perm(a3, a2, 0x05010400), t3_ = __builtin_amdgcn_perm(a3, a2, 0x07030602);
          o[4 * d + 0][m] = __builtin_amdgcn_perm(t2_, t0_, 0x05040100);
          o[4 * d + 1][m] = __builtin_amdgcn_perm(t2_, t0_, 0x07060302);
          o[4 * d + 2][m] = __builtin_amdgcn_perm(t3_, t1_, 0x05040100);
          o[4 * d + 3][m] = __builtin_amdgcn_perm(t3_, t1_, 0x07060302);
        }
      const u32 lw = lds0 + wv * 8192;
      const u32 wbase = lw + (q >> 1) * 1024 + (8 * (q & 1) + 16 * kq) * 16;
#pragma unroll
      for (int e = 0; e < 8; ++e) *(lds_u32x4p)(size_t)(wbase + ((e ^ (q & 7)) << 4)) = o[e];
      waitl0();
      const u32 ra = lw + (((lane & ~7) | ((lane & 7) ^ ((lane >> 3) & 1))) << 4);
      const u32 dst = (call & 1) * SLOT + WREG + ph * RS + cq * 4 * BS + blk * 1024 + lane * 16;
#pragma unroll
      for (int i = 0; i < 8; ++i) o[i] = *(lds_u32x4p)(size_t)((ra ^ ((i & 3) << 5)) + i * 1024);
#pragma unroll
      for (int i = 0; i < 8; ++i) __builtin_amdgcn_raw_buffer_store_b128(o[i], rme, dst + (i >> 1) * BS + (i & 1) * 8192, 0, SYS);
      waitv<0>();
      u32 old = 0;
      if (lane == 0) old = __hip_atomic_fetch_add((glb_u32p)(P.pws[0] + WCNT + stage * 256), 1u, __ATOMIC_RELAXED, __HIP_MEMORY_SCOPE_AGENT);
      old = __builtin_amdgcn_readfirstlane(old);
      if (old + 1 == call * (stage < 2 ? 32u : 192u) && lane < NRANK)
        __hip_atomic_store((glb_u32p)(pw + WFLAG + lane * 1024 + stage * 256), call, __ATOMIC_RELAXED, __HIP_MEMORY_SCOPE_SYSTEM);
    }
    call = __builtin_amdgcn_readfirstlane(callv) + 1;
    if (lane == 0) __builtin_amdgcn_raw_buffer_store_b32(call, rme, ctr, 0, AGT);
    const int gs = p + 1;
    const u32 wf = WFLAG + gs * 1024;
    if (p < 7 && wv < 2) {
      const int c = 2 * g + wv;
      const u32x4 rs = rawrsrc(P.pws[gs], WS_USED);
      const u32 invd = elect_invalidate(P.pws[0], rme, call, lane, true);
      sleep_until(t0, P.first);
      bool ok = true;
      u32 org = t0 + P.minorg;
#pragma unroll 1
      for (int k = 0; k < 2 && ok; ++k) {
        ok = wait_words(rme, k == 0 ? XFLAG + (gs * 16 + c) * 512 + 256 : lane == 1 ? invd : wf, call, t0, lane);
        const u32 cand = now32() + (k == 0 ? P.godelay : 0);
        if ((int)(cand - org) > 0) org = cand;
      }
      if (lane == 0) { u32x2 o = {call, org}; __builtin_amdgcn_raw_buffer_store_b64(o, rme, ORG + (c * 8 + gs) * 256, 0, AGT); }
      int due = 0;
      const int tau = P.tauw;
      const u32 wb = (call & 1) * SLOT + WREG + c * BS;
      u32x4 v = {0, 0, 0, 0};
#pragma unroll 1
      for (int j = 0; j < 8 && ok; ++j) {
        if (j == 1 || j == 2) {
          u32 w;
          while ((w = __builtin_amdgcn_readfirstlane(lds_ld(ldc))) < (u32)j) __builtin_amdgcn_s_sleep(1);
          asm volatile("" ::: "memory");
          if (w > 2) break;
        }
        u32 tab = wb + phase_at(j, rank & 3) * RS + lane * 1024;
        int cnt = 16, extra = j == 7 && P.plan;
#pragma unroll 1
        for (int i = 0; i < cnt + extra; ++i) {
          if (j == 6 || extra) {
            u32 w = __builtin_amdgcn_readfirstlane(lds_ld(ldc + 4 + wv * 4));
            if (j == 6) { if (i + (int)(w & 255) >= 16) break; }
            else {
              while (i == cnt && w == 0) { __builtin_amdgcn_s_sleep(2); w = __builtin_amdgcn_readfirstlane(lds_ld(ldc + 4 + wv * 4)); }
              if (w) {
                asm volatile("" ::: "memory");
                const int m = (w >> 8) & 255;
                const u32 ex = lds_ld(lds0 + 64 * KiB + wv * 256 + ((lane - i) & 63) * 4);
                tab = lane - i < m ? ex : tab - m * 1024;
                cnt += m; extra = 0;
                if (i == cnt) break;
              }
            }
          }
          const u32 vo = __builtin_amdgcn_readlane(tab, i) + lane * 16;
          UNTIL(org, due);
          due += tau;
          waitv<PWIN - 1>();
          ALOAD(v, rs, vo, 0);
        }
      }
      waitv<0>();
      asm volatile("" :: "v"(v));
    } else if (p < 7 && wv == 4) {
#pragma unroll 1
      for (u32 s = 1; s <= 2; ++s) {
        sleep_until(t0, s == 1 ? 600 : 1000);
        const bool ok = wait_words(rme, wf + s * 256, call, t0, lane);
        if (lane == 0) lds_st(ldc, ok ? s : 3);
        if (!ok) break;
      }
    } else if (p < 7 && (wv == 5 || wv == 6) && P.plan) {
      const int c = 2 * g + wv - 5;
      const u32 og = ORG + (c * 8 + gs) * 256;
      sleep_until(t0, P.plan);
      if (wait_words(rme, og, call, t0, lane)) {
        const u32 org = __builtin_amdgcn_readfirstlane(__builtin_amdgcn_raw_buffer_load_b32(rme, og + 4, 0, SYS));
        UNTIL(org, 90 * P.tauw);
        Plan pl;
        do { pl = make_plan(rme, c, call, P.tauw, lane); } while (!pl.good && (int)((now32() - org) << 8) < 140 * P.tauw);
        const int n = __builtin_amdgcn_readlane(pl.n, gs), m = __builtin_amdgcn_readlane(pl.m, gs), u = __builtin_amdgcn_readlane(pl.pc, gs) + lane;
        int a = 0, f = 0;
#pragma unroll
        for (int i = 1; i < 8; ++i) { const int ps = __builtin_amdgcn_readlane(pl.ps, i); const int ni = __builtin_amdgcn_readlane(pl.n, i); if (u >= ps && ni) { a = i; f = MF - ni + u - ps; } }
        const u32 slot = (call & 1) * SLOT;
        const u32 off = slot + MREG + (((a ^ gs) * 4 + (rank & 3)) * 16 + c) * MS + (lane < m ? f : 0) * 1024;
        lds_st(lds0 + 64 * KiB + (wv - 5) * 256 + lane * 4, off);
        waitl0();
        if (lane == 0) lds_st(ldc + 4 + (wv - 5) * 4, n + 256 * m + 65536);
      } else if (lane == 0) lds_st(ldc + 4 + (wv - 5) * 4, 65536);
    } else if (p >= 8 && p < 15 && wv < 2 && P.plan) {
      const int ms = p - 7, c = 2 * g + wv;
      const u32 og = ORG + (c * 8 + ms) * 256;
      const u32 invd = elect_invalidate(P.pws[0], rme, call, lane, true);
      auto rs = mkrsrc(P.pws[ms], WS_USED);
      char* pw = P.pws[lane & 7];
      sleep_until(t0, 1200);
      if (wait_words(rme, lane == 0 ? og : lane == 1 ? invd : WFLAG + ms * 1024 + 512, call, t0, lane)) {
        const u32 org = __builtin_amdgcn_readfirstlane(__builtin_amdgcn_raw_buffer_load_b32(rme, og + 4, 0, SYS));
        const u32 slot = (call & 1) * SLOT;
#pragma unroll 1
        for (int j = 2; j < 5; ++j) {
          const int ph = phase_at(j, rank & 3);
          UNTIL(org, (j * 16 + 16) * P.tauw + P.mmargin * 256);
          u32x4 v[MF];
#pragma unroll
          for (int i = 0; i < MF; ++i) v[i] = __builtin_amdgcn_raw_buffer_load_b128(rs, slot + WREG + ph * RS + c * BS + (16 - MF + i) * 1024 + lane * 16, 0, 0);
#pragma unroll
          for (int i = 0; i < MF; ++i) __builtin_amdgcn_raw_buffer_store_b128(v[i], rme, slot + MREG + ((ms * 4 + ph - 4) * 16 + c) * MS + i * 1024 + lane * 16, 0, SYS);
        }
        waitv<0>();
        u32 old = 0;
        if (lane == 0) old = __hip_atomic_fetch_add((glb_u32p)(P.pws[0] + WCNT + (4 + ms) * 256), 1u, __ATOMIC_RELAXED, __HIP_MEMORY_SCOPE_AGENT);
        old = __builtin_amdgcn_readfirstlane(old);
        if (old + 1 == call * 16u && lane < NRANK && (lane & 3) && lane != ms)
          __hip_atomic_store((glb_u32p)(pw + MFLAG + (lane * 8 + (lane ^ ms)) * 128), call, __ATOMIC_RELAXED, __HIP_MEMORY_SCOPE_SYSTEM);
      }
    }
  } else {
    const u32 t0 = now32();
    {
      u32x4 z = {0, 0, 0, 0};
#pragma unroll
      for (int i = 0; i < 8; ++i) *(lds_u32x4p)(size_t)(lds0 + wv * 8192 + i * 1024 + lane * 16) = z;
      if (wv == 0 && lane < 16) *(lds_u32p)(size_t)(ldc + lane * 4) = 0;
    }
    const u32 call = __builtin_amdgcn_readfirstlane(callv) + 1;
    if (lane == 0) __builtin_amdgcn_raw_buffer_store_b32(call, rme, ctr, 0, AGT);
    const float scale = P.sx[0] * P.sw[0];
    const u32 invd = elect_invalidate(P.pws[0], rme, call, lane, wv == 1);
    waitl0();
    __builtin_amdgcn_s_barrier();
    asm volatile("" ::: "memory");
    const int gs = wv;
    const int c = 2 * g + (cu & 1), rg = cu >> 1;
    const u32 slot = (call & 1) * SLOT;
    const u32 wf = WFLAG + gs * 1024;
    const u32 og = ORG + (c * 8 + gs) * 256;
    bool ok;
    i32x4 a0[16], a1[16], bf[8];
    u32 org;
    {
      u32 off = wf;
      if (gs) {
        sleep_until(t0, P.cfirst);
        off = lane == 0 ? wf : lane == 1 ? og : lane == 2 ? invd : XFLAG + (gs * 16 + 2 * rg + (lane & 1)) * 512;
      }
      ok = wait_words(rme, off, call, t0, lane);
      const u32 ov = __builtin_amdgcn_raw_buffer_load_b32(rme, og + 4, 0, SYS);
      u32 xo = gs ? slot + XREG + gs * RS + 2 * rg * BS + lane * 16 : rank * XB + 2 * rg * 16384 + (lane & 15) * 512 + (lane >> 4) * 16;
      const u32 xs1 = gs ? BS : 16384, xsf = gs ? 1024 : 64;
      auto ra = gs ? rme : mkrsrc(P.x, WB);
#pragma unroll
      for (int f = 0; f < 8; ++f) {
        a0[f] = (i32x4)__builtin_amdgcn_raw_buffer_load_b128(ra, xo, 0, 0);
        a0[8 + f] = (i32x4)__builtin_amdgcn_raw_buffer_load_b128(ra, xo + 8192, 0, 0);
        a1[f] = (i32x4)__builtin_amdgcn_raw_buffer_load_b128(ra, xo + xs1, 0, 0);
        a1[8 + f] = (i32x4)__builtin_amdgcn_raw_buffer_load_b128(ra, xo + xs1 + 8192, 0, 0);
        xo += xsf;
      }
      org = gs && ok ? __builtin_amdgcn_readfirstlane(ov) : now32() - P.lag;
      waitv<0>();
#pragma unroll
      for (int f = 0; f < 16; ++f) asm volatile("" : "+v"(a0[f]), "+v"(a1[f]));
    }
    int due = P.lag * 256;
    const int tau = gs ? P.tauw - (P.lag - P.lagend) * 2 : P.tauself;
    const u64 wb = (u64)P.pws[gs] + slot + WREG + c * BS + (lane & 15) * 1024;
    const u32 l16 = lane * 16;
    int shift = 0, cap = 0x7fffffff;
    auto rout = mkrsrc(P.out, OUT_BYTES);
    u32 lla = lds0 + (56 + gs) * 1024 + ((lane * 16) ^ ((lane >> 5) * 64));
    u32 loa = ((rg * 64 + gs * 8 + (lane >> 3)) * 4096 + 3 * 512 + c * 32 + (lane & 7) * 4) * 4;
    asm volatile("" : "+v"(lla), "+v"(loa));
    bool planned = gs == 0 || !P.plan;
#pragma unroll
    for (int f = 0; f < 8; ++f) bf[f] = i32x4{0, 0, 0, 0};
#pragma unroll 1
    for (int j = 0; j < 8; ++j) {
      if (ok && (j == 1 || j == 2)) ok = wait_words(rme, wf + j * 256, call, t0, lane);
      if (!ok && lane == 0) lds_st(ldc + 32, 1);
      const u32 nrec = ok ? 1024 : 0;
      u64 tab = wb + phase_at(j, rank & 3) * RS;
      if (j >= 6 && !planned && ok) {
        const Plan pl = make_plan(rme, c, call, P.tauw, lane);
        const int n = __builtin_amdgcn_readlane(pl.n, gs), u = __builtin_amdgcn_readlane(pl.ps, gs) + lane - (16 - n);
        int h = 0;
#pragma unroll
        for (int i = 1; i < 8; ++i) if (u >= __builtin_amdgcn_readlane(pl.pc, i) && __builtin_amdgcn_readlane(pl.m, i)) h = i;
        if (j == 6 && lane >= 16 - n && lane < 16) tab = (u64)P.pws[h] + slot + MREG + (((gs ^ h) * 4 + (rank & 3)) * 16 + c) * MS + (lane - (16 - MF)) * 1024;
        shift = (__builtin_amdgcn_readlane(pl.m, gs) - n) * P.tauw;
        if (j == 6) cap = due + (16 - n) * tau;
        planned = pl.good;
      }
      if (j == 7) { due += shift; cap = 0x7fffffff; }
      const int jump = j == 7 ? P.jump * 256 : 0;
      const u32 tlo = (u32)tab, thi = (u32)(tab >> 32) & 0xffffu;
      i32x4 acc[4];
      const u32 ad = lds0 + j * 8192 + (lane >> 4) * 512 + ((lane >> 4) & 1) * 64 + (lane & 15) * 4;
#pragma unroll
      for (int f = 0; f < 16 + DIST; ++f) {
        if (f >= DIST) {
          const int m = f - DIST;
          AWAIT((f < 16 ? DIST - 1 : 15 - m), bf[m & 7]);
          if ((m & 7) == 0) { acc[0] = i32x4{0, 0, 0, 0}; acc[1] = acc[0]; acc[2] = acc[0]; acc[3] = acc[0]; }
          acc[0] = __builtin_amdgcn_mfma_i32_16x16x64_i8(a0[m & 7], bf[m & 7], acc[0], 0, 0, 0);
          acc[1] = __builtin_amdgcn_mfma_i32_16x16x64_i8(a0[8 + (m & 7)], bf[m & 7], acc[1], 0, 0, 0);
          acc[2] = __builtin_amdgcn_mfma_i32_16x16x64_i8(a1[m & 7], bf[m & 7], acc[2], 0, 0, 0);
          acc[3] = __builtin_amdgcn_mfma_i32_16x16x64_i8(a1[8 + (m & 7)], bf[m & 7], acc[3], 0, 0, 0);
          asm volatile("" : "+v"(acc[0]), "+v"(acc[1]), "+v"(acc[2]), "+v"(acc[3]));
          if ((m & 7) == 7) {
#pragma unroll
            for (int e = 0; e < 16; ++e)
              __hip_atomic_fetch_add((lds_u32p)(size_t)((ad ^ ((m >> 3) * 64)) + (e >> 2) * 2048 + (e & 3) * 128), (u32)acc[e >> 2][e & 3], __ATOMIC_RELAXED, __HIP_MEMORY_SCOPE_WORKGROUP);
          }
        }
        if (f < 16) {
          if (f == JF) due -= jump;
          { const int dd = min(due, cap); UNTIL(org, dd); }
          due += tau;
          const u32x4 rf = {(u32)__builtin_amdgcn_readlane(tlo, f), (u32)__builtin_amdgcn_readlane(thi, f), nrec, 0x00020000u};
          asm volatile("s_nop 4\n\tbuffer_load_dwordx4 %0, %1, %2, 0 offen" : "+v"(bf[f & 7]) : "v"(l16), "s"(rf) : "memory");
        }
      }
      waitl0();
      if (j < 7 && lane == 0) __hip_atomic_fetch_add((lds_u32p)(size_t)(ldc + j * 4), 1u, __ATOMIC_RELAXED, __HIP_MEMORY_SCOPE_WORKGROUP);
    }
    auto emit = [&](u32 la, u32 oa) {
      const u32 isbad = lds_ld(ldc + 32);
      const u32x4 v = *(lds_u32x4p)(size_t)la;
      const float sc = isbad ? __builtin_nanf("") : scale;
      const float lo = isbad ? __builtin_nanf("") : 0.f;
      f32x4 y;
#pragma unroll
      for (int e = 0; e < 4; ++e) y[e] = fmaxf((float)(int)v[e] * sc, lo);
      __builtin_amdgcn_raw_buffer_store_b128((u32x4)y, rout, oa, 0, SYS);
    };
    if (__builtin_expect(gs == 0, 0)) {
#pragma unroll 1
      for (int u = 0; u < 56; ++u) {
        const int j = u >> 3;
        while (__builtin_amdgcn_readfirstlane(lds_ld(ldc + j * 4)) != 8) __builtin_amdgcn_s_sleep(1);
        asm volatile("" ::: "memory");
        emit(lla + (u - 56) * 1024, loa + (u & 7) * 8 * 16384 + (phase_at(j, rank & 3) - 3) * 2048);
      }
    }
    __builtin_amdgcn_s_barrier();
    asm volatile("" ::: "memory");
    emit(lla, loa);
  }
}


extern "C" void kernel_launch(int rank, int nranks, ncclComm_t comm, hipStream_t stream, void* const* d_in, const int* in_sizes, int n_in,
                              void* d_out, int out_size, const int* in_dtypes, int out_dtype, void* const* d_out_all,
                              void* const* d_ws_all, size_t ws_bytes) {
  if (nranks != NRANK || (unsigned)rank >= (unsigned)NRANK || ws_bytes < WS_USED || n_in < 4 || in_sizes[0] != (int)WB ||
      in_sizes[1] != (int)WB || out_size != (int)(OUT_BYTES / 4)) {
    fprintf(stderr, "kernel_launch: unexpected problem shape\n");
    return;
  }
  Params P;
  P.x = (const char*)d_in[0]; P.w = (const char*)d_in[1]; P.sx = (const float*)d_in[2]; P.sw = (const float*)d_in[3];
  P.out = (float*)d_out;
  for (int l = 0; l < NRANK; ++l) P.pws[l] = (char*)d_ws_all[rank ^ l];
  P.rank = rank;
  P.taux = 2780 * 256 / 100; P.tauw = 2690 * 256 / 100; P.tauself = 1000 * 256 / 100;
  P.first = 250; P.cfirst = 550; P.minorg = 440; P.godelay = 145;
  P.bdelay = 150; P.cdelay = 600; P.cstag = 30; P.lag = 300; P.lagend = 60;
  P.mmargin = 40; P.plan = 2000; P.jump = 90;
  fused<<<256, 512, 0, stream>>>(P);
}
```
